```python
import numpy as np

kernel_name = "gpu_ag_gemm_m4096_k4096_n4096_bf16_reluqe4m3"


ABSMAX_RTOL = 0.09

COMPUTE_DTYPE = "bfloat16"
EPILOGUE = "relu_quant_e4m3"
IN_QUANT = None
OUT_QUANT = "e4m3"
IN_DTYPES = {"x": "bfloat16", "w_mat": "bfloat16"}
WORLD_SIZE = 8

def _to_bf16(x):
    u = np.ascontiguousarray(x, dtype=np.float32).view(np.uint32)
    return ((u + 0x7FFF + ((u >> 16) & 1)) & 0xFFFF0000).view(np.float32)

def setup_inputs():

    x = _to_bf16(np.random.randn(4096, 4096).astype(np.float32))
    w_mat = _to_bf16((np.random.randn(4096, 4096) / np.sqrt(4096)).astype(np.float32))
    return {"x": x, "w_mat": w_mat}

def _snap_fp8(x, mant_bits, max_abs, min_normal):
    a = np.abs(x.astype(np.float64))
    _m, _e = np.frexp(a)
    q = np.where(a >= min_normal, np.ldexp(1.0, _e - 1 - mant_bits),
                 min_normal * 2.0 ** -mant_bits)
    snapped = np.minimum(np.round(a / q) * q, max_abs)
    return (np.sign(x) * snapped).astype(np.float32)
def _quant_dequant(y, amax):
    scale = amax / 448.0
    q = _snap_fp8(y / scale, 3, 448.0, 2.0 ** -6)
    return (q * scale).astype(np.float32)
def _epilogue(y):
    y = np.maximum(y, 0.0)
    return _quant_dequant(y, np.abs(y).max())

def reference(x, w_mat):
    y = (x @ w_mat).astype(np.float32)
    return _epilogue(y)
```

```cpp
#include <hip/hip_runtime.h>
#include <rccl/rccl.h>
#include <cstdio>
#include <cstdint>

typedef __attribute__((ext_vector_type(8))) short bf16x8;
typedef __attribute__((ext_vector_type(4))) short s16x4;
typedef __attribute__((address_space(3))) s16x4 lds_s16x4;
typedef __attribute__((ext_vector_type(4))) float f32x4;
typedef __attribute__((ext_vector_type(4))) unsigned u32x4;

#define HIP_CHECK(expr) do { hipError_t _e = (expr); if (_e != hipSuccess) { \
    fprintf(stderr, "HIP error %s at %s:%d\n", hipGetErrorString(_e), __FILE__, __LINE__); \
    return; } } while (0)

namespace {

constexpr int NR = 8;
constexpr int MM = 4096, KK = 4096;
constexpr int NSH = 512;
constexpr int MSH = 512;
constexpr int BM = 128, BN = 64, BK = 64;
constexpr int KS = 256;
constexpr int NSL = KK / KS;
constexpr int TPS = KS / BK;
constexpr int NT = KK / BK;
constexpr int PP = 8;
constexpr int NPUSH = 7 * PP;
constexpr int NG = (MM / BM) * (NSH / BN);
constexpr int NWG = NPUSH + NG;

constexpr size_t OFF_CNT = 0;
constexpr size_t OFF_TMO = 128;
constexpr size_t OFF_FAN = 256;
constexpr size_t OFF_LMAX = 384;
constexpr size_t OFF_AMAX = 1024;
constexpr size_t OFF_FLAG = 4096;
constexpr size_t OFF_STAGE = 1u << 20;
constexpr size_t SLOT = (size_t)MSH * KK * 2;
constexpr size_t WS_NEED = OFF_STAGE + NR * SLOT;

constexpr unsigned long long DEADLINE = 50000000ull;

constexpr int A_BYTES = BM * BK * 2;
constexpr int B_BYTES = BK * BN * 2;
constexpr int BUF_BYTES = A_BYTES + B_BYTES;
constexpr int LDS_BYTES = 2 * BUF_BYTES + 64;

struct Args {
  const unsigned short* x;
  const unsigned short* w;
  unsigned char* out;
  char* ws[NR];
  int rank;
};

__device__ __forceinline__ int lds_byte(int r, int c) {
  int st = (r >> 4) * 2 + (c >> 5), rr = r & 15, cc = c & 31, ob = rr * 64 + cc * 2;
  return st * 1024 + (ob ^ (((ob >> 9) & 1) << 5));
}
__device__ __forceinline__ void stage_rc(int b, int& R, int& C) {
  int st = b / 1024, sb = b % 1024, swz = sb ^ (((sb >> 9) & 1) << 5);
  R = (st >> 1) * 16 + swz / 64; C = (st & 1) * 32 + (swz % 64) / 2;
}
__device__ __forceinline__ int bswz(int row) { return ((row >> 1) & 1) | (((row >> 3) & 1) << 1); }

__device__ __forceinline__ bool wait_u32(const unsigned* flag, unsigned target, unsigned* tmo, unsigned call) {
  unsigned long long t0 = __builtin_amdgcn_s_memrealtime();
  while (__hip_atomic_load(flag, __ATOMIC_RELAXED, __HIP_MEMORY_SCOPE_SYSTEM) < target) {
    if (__hip_atomic_load(tmo, __ATOMIC_RELAXED, __HIP_MEMORY_SCOPE_AGENT) == call ||
        __builtin_amdgcn_s_memrealtime() - t0 > DEADLINE) {
      __hip_atomic_store(tmo, call, __ATOMIC_RELAXED, __HIP_MEMORY_SCOPE_AGENT);
      return false;
    }
    __builtin_amdgcn_s_sleep(2);
  }
  return true;
}

__global__ __launch_bounds__(256, 2) void ag_gemm_q(Args a) {
  extern __shared__ __attribute__((aligned(16))) char smem[];
  const int tid = threadIdx.x;
  const int rank = a.rank;
  char* myws = a.ws[rank];
  unsigned* cnt = (unsigned*)(myws + OFF_CNT);
  unsigned* tmo = (unsigned*)(myws + OFF_TMO);
  const unsigned call = __hip_atomic_load(cnt, __ATOMIC_RELAXED, __HIP_MEMORY_SCOPE_AGENT) / NWG + 1;

  if (blockIdx.x < NPUSH) {
    const int pi = blockIdx.x % 7, part = blockIdx.x / 7;
    const int p = (rank + 1 + pi) & (NR - 1);
    char* dst = a.ws[p] + OFF_STAGE + (size_t)rank * SLOT;
    const char* src = (const char*)a.x;
    char* flags = a.ws[p] + OFF_FLAG + (size_t)rank * NSL * 128;
    constexpr int RP = MSH / PP;
    constexpr int LPR = KS * 2 / 16;
    constexpr int RPI = 256 / LPR;
    constexpr int NIT = RP / RPI;
    const int lr = tid % LPR, r0 = tid / LPR;
    for (int s = 0; s < NSL; ++s) {
      u32x4 v[NIT];
#pragma unroll
      for (int i = 0; i < NIT; ++i) {
        size_t off = (size_t)(part * RP + i * RPI + r0) * (KK * 2) + (size_t)s * (KS * 2) + lr * 16;
        v[i] = *(const u32x4*)(src + off);
      }
#pragma unroll
      for (int i = 0; i < NIT; ++i) {
        size_t off = (size_t)(part * RP + i * RPI + r0) * (KK * 2) + (size_t)s * (KS * 2) + lr * 16;
        *(u32x4*)(dst + off) = v[i];
      }
      asm volatile("s_waitcnt vmcnt(0)" ::: "memory");
      __syncthreads();
      if (tid == 0) {
        __builtin_amdgcn_fence(__ATOMIC_RELEASE, "");
        asm volatile("s_waitcnt vmcnt(0)" ::: "memory");
        __hip_atomic_fetch_add((unsigned*)(flags + s * 128), 1u, __ATOMIC_RELAXED, __HIP_MEMORY_SCOPE_SYSTEM);
      }
    }
    if (tid == 0) __hip_atomic_fetch_add(cnt, 1u, __ATOMIC_RELAXED, __HIP_MEMORY_SCOPE_AGENT);
    return;
  }

  const int g = blockIdx.x - NPUSH;
  const int xcd = g & 7, idx = g >> 3;
  const int trow = xcd + 8 * (idx >> 3), tcol = idx & 7;
  const int brow = trow * BM, bcol = tcol * BN;
  const int srcr = brow / MSH;
  const bool remote = (srcr != rank);
  const unsigned short* Abase = remote ? (const unsigned short*)(myws + OFF_STAGE + (size_t)srcr * SLOT)
                                       : a.x;
  const int arow0 = brow % MSH;
  const unsigned* flags = (const unsigned*)(myws + OFF_FLAG + (size_t)srcr * NSL * 128);
  const unsigned ftarget = call * PP;
  int* sflag = (int*)(smem + 2 * BUF_BYTES);

  const int wid = tid >> 6, lane = tid & 63, wr = wid >> 1, wc = wid & 1, fr = lane & 15, fq = lane >> 4;

  const unsigned short* asrc[4];
#pragma unroll
  for (int i = 0; i < 4; ++i) {
    int R, C; stage_rc(tid * 16 + i * 4096, R, C);
    asrc[i] = Abase + (size_t)(arow0 + R) * KK + C;
  }
  const unsigned short* bsrc[2];
#pragma unroll
  for (int i = 0; i < 2; ++i) {
    int b = tid * 16 + i * 4096, row = b >> 7, sl = (b & 127) >> 4;
    int col = (((sl >> 1) ^ bswz(row)) << 4) + ((sl & 1) << 3);
    bsrc[i] = a.w + (size_t)row * NSH + bcol + col;
  }
  auto stage = [&](int t, int buf) {
    char* sa = smem + buf * BUF_BYTES;
    char* sb = sa + A_BYTES;
#pragma unroll
    for (int i = 0; i < 4; ++i)
      __builtin_amdgcn_global_load_lds((const unsigned*)(asrc[i] + (size_t)t * BK), (unsigned*)(sa + tid * 16 + i * 4096), 16, 0, 0);
#pragma unroll
    for (int i = 0; i < 2; ++i)
      __builtin_amdgcn_global_load_lds((const unsigned*)(bsrc[i] + (size_t)t * BK * NSH), (unsigned*)(sb + tid * 16 + i * 4096), 16, 0, 0);
  };
  auto wait_slice = [&](int s) {
    if (remote) {
      if (tid == 0) {
        wait_u32(flags + s * 32, ftarget, tmo, call);
        __builtin_amdgcn_fence(__ATOMIC_ACQUIRE, "");
        asm volatile("s_waitcnt vmcnt(0)" ::: "memory");
      }
      __syncthreads();
    }
  };

  int aoff[4][2], boff[2][2][2];
#pragma unroll
  for (int m = 0; m < 4; ++m)
#pragma unroll
    for (int kk = 0; kk < 2; ++kk) aoff[m][kk] = lds_byte(wr * 64 + m * 16 + fr, kk * 32 + fq * 8);
#pragma unroll
  for (int n = 0; n < 2; ++n)
#pragma unroll
    for (int kk = 0; kk < 2; ++kk)
#pragma unroll
      for (int t = 0; t < 2; ++t) {
        int row = kk * 32 + 8 * fq + 4 * t + (fr >> 2);
        boff[n][kk][t] = A_BYTES + row * 128 + (((wc * 2 + n) ^ bswz(row)) << 5) + ((lane & 3) << 3);
      }

  f32x4 acc[4][2];
#pragma unroll
  for (int m = 0; m < 4; ++m)
#pragma unroll
    for (int n = 0; n < 2; ++n) acc[m][n] = f32x4{0.f, 0.f, 0.f, 0.f};

  wait_slice(0);
  stage(0, 0);
  for (int t = 0; t < NT; ++t) {
    const int buf = t & 1;
    asm volatile("s_waitcnt vmcnt(0)" ::: "memory");
    __syncthreads();
    if (t + 1 < NT) {
      if ((t + 1) % TPS == 0) wait_slice((t + 1) / TPS);
      stage(t + 1, buf ^ 1);
    }
    const char* sbuf = smem + buf * BUF_BYTES;
    bf16x8 af[4][2], bfr[2][2];
#pragma unroll
    for (int m = 0; m < 4; ++m)
#pragma unroll
      for (int kk = 0; kk < 2; ++kk) af[m][kk] = *(const bf16x8*)(sbuf + aoff[m][kk]);
#pragma unroll
    for (int n = 0; n < 2; ++n)
#pragma unroll
      for (int kk = 0; kk < 2; ++kk) {
        s16x4 lo = __builtin_amdgcn_ds_read_tr16_b64_v4i16((lds_s16x4*)(sbuf + boff[n][kk][0]));
        s16x4 hi = __builtin_amdgcn_ds_read_tr16_b64_v4i16((lds_s16x4*)(sbuf + boff[n][kk][1]));
        bfr[n][kk] = bf16x8{lo[0], lo[1], lo[2], lo[3], hi[0], hi[1], hi[2], hi[3]};
      }
#pragma unroll
    for (int kk = 0; kk < 2; ++kk)
#pragma unroll
      for (int m = 0; m < 4; ++m)
#pragma unroll
        for (int n = 0; n < 2; ++n)
          acc[m][n] = __builtin_amdgcn_mfma_f32_16x16x32_bf16(af[m][kk], bfr[n][kk], acc[m][n], 0, 0, 0);
  }

  float mx = 0.f;
#pragma unroll
  for (int m = 0; m < 4; ++m)
#pragma unroll
    for (int n = 0; n < 2; ++n)
#pragma unroll
      for (int j = 0; j < 4; ++j) { float v = fmaxf(acc[m][n][j], 0.f); acc[m][n][j] = v; mx = fmaxf(mx, v); }
#pragma unroll
  for (int o = 32; o >= 1; o >>= 1) mx = fmaxf(mx, __shfl_xor(mx, o));
  __syncthreads();
  float* sred = (float*)(smem + 2 * BUF_BYTES + 16);
  if (lane == 0) sred[wid] = mx;
  __syncthreads();
  if (tid == 0) {
    float m4 = fmaxf(fmaxf(sred[0], sred[1]), fmaxf(sred[2], sred[3]));
    unsigned long long key = ((unsigned long long)call << 32) | __float_as_uint(m4);
    unsigned long long* lmax = (unsigned long long*)(myws + OFF_LMAX);
    __hip_atomic_fetch_max(lmax, key, __ATOMIC_RELAXED, __HIP_MEMORY_SCOPE_AGENT);
    asm volatile("s_waitcnt vmcnt(0)" ::: "memory");
    unsigned tk = __hip_atomic_fetch_add((unsigned*)(myws + OFF_FAN), 1u, __ATOMIC_ACQ_REL, __HIP_MEMORY_SCOPE_AGENT);
    if (tk == call * NG - 1) {
      unsigned long long fin = __hip_atomic_load(lmax, __ATOMIC_RELAXED, __HIP_MEMORY_SCOPE_AGENT);
      for (int r = 0; r < NR; ++r)
        __hip_atomic_store((unsigned long long*)(a.ws[r] + OFF_AMAX + rank * 128), fin, __ATOMIC_RELAXED,
                           __HIP_MEMORY_SCOPE_SYSTEM);
    }
  }
  if (wid == 0) {
    float gm = 0.f;
    bool ok = true;
    if (lane < NR) {
      const unsigned long long* slot = (const unsigned long long*)(myws + OFF_AMAX + lane * 128);
      unsigned long long t0 = __builtin_amdgcn_s_memrealtime();
      unsigned long long v;
      while (((v = __hip_atomic_load(slot, __ATOMIC_RELAXED, __HIP_MEMORY_SCOPE_SYSTEM)) >> 32) < call) {
        if (__hip_atomic_load(tmo, __ATOMIC_RELAXED, __HIP_MEMORY_SCOPE_AGENT) == call ||
            __builtin_amdgcn_s_memrealtime() - t0 > DEADLINE) {
          __hip_atomic_store(tmo, call, __ATOMIC_RELAXED, __HIP_MEMORY_SCOPE_AGENT);
          ok = false;
          break;
        }
        __builtin_amdgcn_s_sleep(1);
      }
      gm = ok ? __uint_as_float((unsigned)v) : 0.f;
    }
#pragma unroll
    for (int o = 4; o >= 1; o >>= 1) gm = fmaxf(gm, __shfl_xor(gm, o));
    if (lane == 0) {
      bool bad = __hip_atomic_load(tmo, __ATOMIC_RELAXED, __HIP_MEMORY_SCOPE_AGENT) == call;
      sred[4] = gm;
      sflag[0] = bad ? 1 : 0;
    }
  }
  __syncthreads();
  const float gmax = sred[4];
  const bool bad = sflag[0] != 0;
  const float scale = gmax / 448.f;

  constexpr int QS = 80;
  unsigned char* q = (unsigned char*)smem;
#pragma unroll
  for (int m = 0; m < 4; ++m)
#pragma unroll
    for (int n = 0; n < 2; ++n) {
      float v0 = acc[m][n][0] / scale, v1 = acc[m][n][1] / scale, v2 = acc[m][n][2] / scale, v3 = acc[m][n][3] / scale;
      v0 = fminf(v0, 448.f); v1 = fminf(v1, 448.f); v2 = fminf(v2, 448.f); v3 = fminf(v3, 448.f);
      int p01 = __builtin_amdgcn_cvt_pk_fp8_f32(v0, v1, 0, false);
      int p23 = __builtin_amdgcn_cvt_pk_fp8_f32(v2, v3, 0, false);
      int r = wr * 64 + m * 16 + fq * 4, c = wc * 32 + n * 16 + fr;
      q[(r + 0) * QS + c] = (unsigned char)(p01 & 0xff);
      q[(r + 1) * QS + c] = (unsigned char)((p01 >> 8) & 0xff);
      q[(r + 2) * QS + c] = (unsigned char)(p23 & 0xff);
      q[(r + 3) * QS + c] = (unsigned char)((p23 >> 8) & 0xff);
    }
  __syncthreads();
#pragma unroll
  for (int i = 0; i < 2; ++i) {
    int r = (tid >> 2) + i * 64, sg = tid & 3;
    u32x4 v = *(const u32x4*)(q + r * QS + sg * 16);
    if (bad) v = u32x4{0x7f7f7f7fu, 0x7f7f7f7fu, 0x7f7f7f7fu, 0x7f7f7f7fu};
    *(u32x4*)(a.out + (size_t)(brow + r) * NSH + bcol + sg * 16) = v;
  }
  if (tid == 0) __hip_atomic_fetch_add(cnt, 1u, __ATOMIC_RELAXED, __HIP_MEMORY_SCOPE_AGENT);
}

}

extern "C" void kernel_launch(int rank, int nranks, ncclComm_t comm, hipStream_t stream,
                              void* const* d_in, const int* in_sizes, int n_in, void* d_out, int out_size,
                              const int* in_dtypes, int out_dtype, void* const* d_out_all,
                              void* const* d_ws_all, size_t ws_bytes) {
  if (nranks != NR || ws_bytes < WS_NEED || (unsigned)rank >= (unsigned)NR) {
    fprintf(stderr, "unexpected configuration\n");
    return;
  }
  Args a;
  a.x = (const unsigned short*)d_in[0];
  a.w = (const unsigned short*)d_in[1];
  a.out = (unsigned char*)d_out;
  for (int r = 0; r < NR; ++r) a.ws[r] = (char*)d_ws_all[r];
  a.rank = rank;
  hipLaunchKernelGGL(ag_gemm_q, dim3(NWG), dim3(256), LDS_BYTES, stream, a);
  HIP_CHECK(hipGetLastError());
}
```

```cpp
#include <hip/hip_runtime.h>
#include <rccl/rccl.h>
#include <cstdio>
#include <cstdint>

typedef __attribute__((ext_vector_type(8))) short bf16x8;
typedef __attribute__((ext_vector_type(4))) short s16x4;
typedef __attribute__((ext_vector_type(4))) float f32x4;
typedef __attribute__((ext_vector_type(4))) unsigned u32x4;

#define HIP_CHECK(expr) do { hipError_t _e = (expr); if (_e != hipSuccess) { \
    fprintf(stderr, "HIP error %s at %s:%d\n", hipGetErrorString(_e), __FILE__, __LINE__); \
    return; } } while (0)

namespace {

#define KS_ 256
#define NSTAGE_ 5
constexpr int NR = 8;
constexpr int MM = 4096, KK = 4096;
constexpr int NSH = 512;
constexpr int MSH = 512;
constexpr int BM = 128, BN = 64, BK = 64;
constexpr int NTHR = 512;
constexpr int KS = KS_;
constexpr int NSTAGE = NSTAGE_;
constexpr int NSL = KK / KS;
constexpr int TPS = KS / BK;
constexpr int NT = KK / BK;
constexpr int PP = 4;
constexpr int NPUSH = 7 * PP;
constexpr int NLOC = (MSH / BM) * (NSH / BN);
constexpr int NG = 7 * NLOC;
constexpr int NWG = NPUSH + NG;

constexpr size_t OFF_CNT = 0;
constexpr size_t OFF_TMO = 128;
constexpr size_t OFF_FAN = 256;
constexpr size_t OFF_LMAX = 384;
constexpr size_t OFF_AMAX = 1024;
constexpr size_t OFF_FLAG = 4096;
constexpr size_t OFF_STAGE = 1u << 20;
constexpr size_t SLOT = (size_t)MSH * KK * 2;
constexpr size_t WS_NEED = OFF_STAGE + NR * SLOT;

#define TS(i) do {} while (0)
constexpr unsigned long long DEADLINE = 50000000ull;

constexpr int A_BYTES = BM * BK * 2;
constexpr int B_BYTES = BK * BN * 2;
constexpr int BUF_BYTES = A_BYTES + B_BYTES;
constexpr int LDS_BYTES = NSTAGE * BUF_BYTES + 64;
static_assert(LDS_BYTES > 81920 && LDS_BYTES <= 163840, "one workgroup per CU");

struct Args {
  const unsigned short* x;
  const unsigned short* w;
  unsigned char* out;
  char* ws[NR];
  int rank;
};

__device__ __forceinline__ int lds_byte(int r, int c) {
  int st = (r >> 4) * 2 + (c >> 5), rr = r & 15, cc = c & 31, ob = rr * 64 + cc * 2;
  return st * 1024 + (ob ^ (((ob >> 9) & 1) << 5));
}
__device__ __forceinline__ void stage_rc(int b, int& R, int& C) {
  int st = b / 1024, sb = b % 1024, swz = sb ^ (((sb >> 9) & 1) << 5);
  R = (st >> 1) * 16 + swz / 64; C = (st & 1) * 32 + (swz % 64) / 2;
}
__device__ __forceinline__ int bswz(int row) { return ((row >> 1) & 1) | (((row >> 3) & 1) << 1); }

__device__ __forceinline__ void wait_u32(const unsigned* flag, unsigned target, unsigned* tmo, unsigned call) {
  unsigned long long t0 = __builtin_amdgcn_s_memrealtime();
  while (__hip_atomic_load(flag, __ATOMIC_RELAXED, __HIP_MEMORY_SCOPE_SYSTEM) < target) {
    if (__hip_atomic_load(tmo, __ATOMIC_RELAXED, __HIP_MEMORY_SCOPE_AGENT) == call ||
        __builtin_amdgcn_s_memrealtime() - t0 > DEADLINE) {
      __hip_atomic_store(tmo, call, __ATOMIC_RELAXED, __HIP_MEMORY_SCOPE_AGENT);
      return;
    }
    __builtin_amdgcn_s_sleep(2);
  }
}

struct GemmCtx {
  char* smem; char* myws;
  unsigned lds0;
  int aoffs[2];
  int boffs;
  int aoff[2][2], boff[2][2][2];
  const unsigned short* w;
  unsigned* tmo; unsigned call;
};

template <bool REMOTE>
__device__ __forceinline__ void gemm_tile(const GemmCtx& c, const unsigned short* Abase, int bcol,
                                          const unsigned* flags, f32x4 (&acc)[2][2]) {
  const int tid = threadIdx.x;
  char* myws = c.myws;
  const unsigned ftarget = c.call * (PP * (NTHR / 64));
  const unsigned short* a0 = Abase + c.aoffs[0];
  const unsigned short* a1 = Abase + c.aoffs[1];
  const unsigned short* b0 = c.w + c.boffs + bcol;
  auto stage = [&](int t, int buf) {
    char* sa = c.smem + buf * BUF_BYTES;
    __builtin_amdgcn_global_load_lds((const unsigned*)(a0 + (size_t)t * BK), (unsigned*)(sa + tid * 16), 16, 0, REMOTE ? 17 : 0);
    __builtin_amdgcn_global_load_lds((const unsigned*)(a1 + (size_t)t * BK), (unsigned*)(sa + tid * 16 + 8192), 16, 0, REMOTE ? 17 : 0);
    __builtin_amdgcn_global_load_lds((const unsigned*)(b0 + (size_t)t * BK * NSH), (unsigned*)(sa + A_BYTES + tid * 16), 16, 0, 0);
  };
  auto wait_slice = [&](int s) {
    if (REMOTE) {
      if (tid == 0) wait_u32(flags + s * 32, ftarget, c.tmo, c.call);
      __builtin_amdgcn_s_barrier();
      TS(1 + s);
    }
  };
#pragma unroll
  for (int m = 0; m < 2; ++m)
#pragma unroll
    for (int n = 0; n < 2; ++n) acc[m][n] = f32x4{0.f, 0.f, 0.f, 0.f};
  __builtin_amdgcn_s_barrier();
#pragma unroll
  for (int t = 0; t < NSTAGE - 1; ++t) {
    if (t % TPS == 0) wait_slice(t / TPS);
    stage(t, t);
  }
  int buf = 0;
  for (int t = 0; t < NT; ++t) {
    if (t + NSTAGE - 2 < NT) asm volatile("s_waitcnt vmcnt(%0)" :: "n"(3 * (NSTAGE - 2)) : "memory");
    else asm volatile("s_waitcnt vmcnt(0)" ::: "memory");
    __builtin_amdgcn_s_barrier();
    if (t + NSTAGE - 1 < NT) {
      if ((t + NSTAGE - 1) % TPS == 0) wait_slice((t + NSTAGE - 1) / TPS);
      stage(t + NSTAGE - 1, buf == 0 ? NSTAGE - 1 : buf - 1);
    }
    const char* sbuf = c.smem + buf * BUF_BYTES;
    const unsigned lbase = c.lds0 + buf * BUF_BYTES;
    bf16x8 af[2][2], bfr[2][2];
    s16x4 bl[2][2][2];
#pragma unroll
    for (int n = 0; n < 2; ++n)
#pragma unroll
      for (int kk = 0; kk < 2; ++kk)
#pragma unroll
        for (int h = 0; h < 2; ++h)
          asm volatile("ds_read_b64_tr_b16 %0, %1" : "=v"(bl[n][kk][h]) : "v"(lbase + c.boff[n][kk][h]) : "memory");
#pragma unroll
    for (int m = 0; m < 2; ++m)
#pragma unroll
      for (int kk = 0; kk < 2; ++kk) af[m][kk] = *(const bf16x8*)(sbuf + c.aoff[m][kk]);
    asm volatile("s_waitcnt lgkmcnt(0)" : "+v"(bl[0][0][0]), "+v"(bl[0][0][1]), "+v"(bl[0][1][0]), "+v"(bl[0][1][1]),
                 "+v"(bl[1][0][0]), "+v"(bl[1][0][1]), "+v"(bl[1][1][0]), "+v"(bl[1][1][1]) :: "memory");
    __builtin_amdgcn_sched_barrier(0);
#pragma unroll
    for (int n = 0; n < 2; ++n)
#pragma unroll
      for (int kk = 0; kk < 2; ++kk) {
        s16x4 lo = bl[n][kk][0], hi = bl[n][kk][1];
        bfr[n][kk] = bf16x8{lo[0], lo[1], lo[2], lo[3], hi[0], hi[1], hi[2], hi[3]};
      }
#pragma unroll
    for (int kk = 0; kk < 2; ++kk)
#pragma unroll
      for (int m = 0; m < 2; ++m)
#pragma unroll
        for (int n = 0; n < 2; ++n)
          acc[m][n] = __builtin_amdgcn_mfma_f32_16x16x32_bf16(af[m][kk], bfr[n][kk], acc[m][n], 0, 0, 0);
    buf = (buf + 1 == NSTAGE) ? 0 : buf + 1;
  }
}

__device__ __forceinline__ float relu_max(f32x4 (&acc)[2][2], float mx) {
#pragma unroll
  for (int m = 0; m < 2; ++m)
#pragma unroll
    for (int n = 0; n < 2; ++n)
#pragma unroll
      for (int j = 0; j < 4; ++j) { float v = fmaxf(acc[m][n][j], 0.f); acc[m][n][j] = v; mx = fmaxf(mx, v); }
  return mx;
}

constexpr int QS = 80;
__device__ __forceinline__ void quant_to_lds(const f32x4 (&acc)[2][2], float scale, unsigned char* q,
                                             int wr, int wc, int fr, int fq) {
#pragma unroll
  for (int m = 0; m < 2; ++m)
#pragma unroll
    for (int n = 0; n < 2; ++n) {
      float v0 = acc[m][n][0] / scale, v1 = acc[m][n][1] / scale, v2 = acc[m][n][2] / scale, v3 = acc[m][n][3] / scale;
      v0 = fminf(v0, 448.f); v1 = fminf(v1, 448.f); v2 = fminf(v2, 448.f); v3 = fminf(v3, 448.f);
      int p01 = __builtin_amdgcn_cvt_pk_fp8_f32(v0, v1, 0, false);
      int p23 = __builtin_amdgcn_cvt_pk_fp8_f32(v2, v3, 0, false);
      int r = wr * 32 + m * 16 + fq * 4, cc = wc * 32 + n * 16 + fr;
      q[(r + 0) * QS + cc] = (unsigned char)(p01 & 0xff);
      q[(r + 1) * QS + cc] = (unsigned char)((p01 >> 8) & 0xff);
      q[(r + 2) * QS + cc] = (unsigned char)(p23 & 0xff);
      q[(r + 3) * QS + cc] = (unsigned char)((p23 >> 8) & 0xff);
    }
}

__global__ __launch_bounds__(NTHR, 2) void ag_gemm_q(Args a) {
  extern __shared__ __attribute__((aligned(16))) char smem[];
  const int tid = threadIdx.x;
  const int rank = a.rank;
  char* myws = a.ws[rank];
  unsigned* cnt = (unsigned*)(myws + OFF_CNT);
  unsigned* tmo = (unsigned*)(myws + OFF_TMO);
  const unsigned call = __hip_atomic_load(cnt, __ATOMIC_RELAXED, __HIP_MEMORY_SCOPE_AGENT) / NWG + 1;

  TS(0);
  if (blockIdx.x < NPUSH) {
    const int pi = blockIdx.x % 7, part = blockIdx.x / 7;
    const int p = (rank + 1 + pi) & (NR - 1);
    char* dst = a.ws[p] + OFF_STAGE + (size_t)rank * SLOT;
    const char* src = (const char*)a.x;
    char* flags = a.ws[p] + OFF_FLAG + (size_t)rank * NSL * 128;
    constexpr int RP = MSH / PP;
    constexpr int LPR = KS * 2 / 16;
    constexpr int RPI = NTHR / LPR;
    constexpr int NIT = RP / RPI;
    static_assert(NIT == 16 || NIT == 8 || NIT == 4 || NIT == 2, "NIT");
    static_assert(NSL % 2 == 0, "NSL");
    const int lr = tid % LPR, r0 = tid / LPR;
    const size_t off0 = (size_t)(part * RP + r0) * (KK * 2) + lr * 16;
    u32x4 va[NIT], vb[NIT];
    auto ld = [&](u32x4 (&d)[NIT], int sl) {
#pragma unroll
      for (int i = 0; i < NIT; ++i) {
        const char* sp = src + off0 + (size_t)i * RPI * (KK * 2) + (size_t)sl * (KS * 2);
        asm volatile("global_load_dwordx4 %0, %1, off" : "=v"(d[i]) : "v"(sp) : "memory");
      }
    };
    auto body = [&](int sl, u32x4 (&cur)[NIT], u32x4 (&nxt)[NIT]) {
      ld(nxt, (sl + 1 < NSL) ? sl + 1 : sl);
      if (sl == 0) asm volatile("s_waitcnt vmcnt(%0)" :: "n"(NIT) : "memory");
      else asm volatile("s_waitcnt vmcnt(%0)" :: "n"(2 * NIT) : "memory");
#pragma unroll
      for (int i = 0; i < NIT; ++i) asm volatile("" : "+v"(cur[i]) :: "memory");
#pragma unroll
      for (int i = 0; i < NIT; ++i) {
        char* dp = dst + off0 + (size_t)i * RPI * (KK * 2) + (size_t)sl * (KS * 2);
        asm volatile("global_store_dwordx4 %0, %1, off sc0 sc1\n\ts_nop 1" :: "v"(dp), "v"(cur[i]) : "memory");
      }
      if (sl > 0) {
        asm volatile("s_waitcnt vmcnt(%0)" :: "n"(2 * NIT) : "memory");
        if ((tid & 63) == 0)
          __hip_atomic_fetch_add((unsigned*)(flags + (sl - 1) * 128), 1u, __ATOMIC_RELAXED, __HIP_MEMORY_SCOPE_SYSTEM);
        TS(sl);
      }
    };
    ld(va, 0);
    for (int sl = 0; sl < NSL; sl += 2) { body(sl, va, vb); body(sl + 1, vb, va); }
    asm volatile("s_waitcnt vmcnt(0)" ::: "memory");
    if ((tid & 63) == 0)
      __hip_atomic_fetch_add((unsigned*)(flags + (NSL - 1) * 128), 1u, __ATOMIC_RELAXED, __HIP_MEMORY_SCOPE_SYSTEM);
    TS(NSL);
    if (tid == 0) __hip_atomic_fetch_add(cnt, 1u, __ATOMIC_RELAXED, __HIP_MEMORY_SCOPE_AGENT);
    return;
  }

  const int g = blockIdx.x - NPUSH;
  const int pair = (g & 7) * (NG / 8) + (g >> 3);
  const int rt = pair >> 3, tcol = pair & 7;
  const int srcr = (rank + 1 + (rt >> 2)) & (NR - 1);
  const int arowR = (rt & 3) * BM;
  const int bcolR = tcol * BN;
  const bool hasloc = g < NLOC;
  const int arowL = (g >> 3) * BM, bcolL = (g & 7) * BN;

  const int wid = tid >> 6, lane = tid & 63, wr = wid >> 1, wc = wid & 1, fr = lane & 15, fq = lane >> 4;
  GemmCtx c;
  c.smem = smem; c.myws = myws; c.lds0 = (unsigned)(size_t)smem; c.w = a.w; c.tmo = tmo; c.call = call;
#pragma unroll
  for (int i = 0; i < 2; ++i) { int R, C; stage_rc(tid * 16 + i * 8192, R, C); c.aoffs[i] = R * KK + C; }
  { int b = tid * 16, row = b >> 7, sl = (b & 127) >> 4;
    c.boffs = row * NSH + (((sl >> 1) ^ bswz(row)) << 4) + ((sl & 1) << 3); }
#pragma unroll
  for (int m = 0; m < 2; ++m)
#pragma unroll
    for (int kk = 0; kk < 2; ++kk) c.aoff[m][kk] = lds_byte(wr * 32 + m * 16 + fr, kk * 32 + fq * 8);
#pragma unroll
  for (int n = 0; n < 2; ++n)
#pragma unroll
    for (int kk = 0; kk < 2; ++kk)
#pragma unroll
      for (int t = 0; t < 2; ++t) {
        int row = kk * 32 + 8 * fq + 4 * t + (fr >> 2);
        c.boff[n][kk][t] = A_BYTES + row * 128 + (((wc * 2 + n) ^ bswz(row)) << 5) + ((lane & 3) << 3);
      }

  f32x4 accL[2][2], accR[2][2];
  float mx = 0.f;
  if (hasloc) {
    gemm_tile<false>(c, a.x + (size_t)arowL * KK, bcolL, nullptr, accL);
    mx = relu_max(accL, mx);
    TS(59);
  }
  gemm_tile<true>(c, (const unsigned short*)(myws + OFF_STAGE + (size_t)srcr * SLOT) + (size_t)arowR * KK, bcolR,
                  (const unsigned*)(myws + OFF_FLAG + (size_t)srcr * NSL * 128), accR);
  mx = relu_max(accR, mx);
  TS(60);

#pragma unroll
  for (int o = 32; o >= 1; o >>= 1) mx = fmaxf(mx, __shfl_xor(mx, o));
  float* sred = (float*)(smem + NSTAGE * BUF_BYTES);
  if (lane == 0) sred[wid] = mx;
  __syncthreads();
  if (tid == 0) {
    float m8 = 0.f;
#pragma unroll
    for (int i = 0; i < 8; ++i) m8 = fmaxf(m8, sred[i]);
    unsigned long long key = ((unsigned long long)call << 32) | __float_as_uint(m8);
    unsigned long long* lmax = (unsigned long long*)(myws + OFF_LMAX);
    __hip_atomic_fetch_max(lmax, key, __ATOMIC_RELAXED, __HIP_MEMORY_SCOPE_AGENT);
    asm volatile("s_waitcnt vmcnt(0)" ::: "memory");
    unsigned tk = __hip_atomic_fetch_add((unsigned*)(myws + OFF_FAN), 1u, __ATOMIC_RELAXED, __HIP_MEMORY_SCOPE_AGENT);
    if (tk == call * NG - 1) {
      unsigned long long fin = __hip_atomic_load(lmax, __ATOMIC_RELAXED, __HIP_MEMORY_SCOPE_AGENT);
      for (int r = 0; r < NR; ++r)
        __hip_atomic_store((unsigned long long*)(a.ws[r] + OFF_AMAX + rank * 128), fin, __ATOMIC_RELAXED,
                           __HIP_MEMORY_SCOPE_SYSTEM);
    }
  }
  TS(61);
  if (wid == 0) {
    float gm = 0.f;
    bool ok = true;
    if (lane < NR) {
      const unsigned long long* slot = (const unsigned long long*)(myws + OFF_AMAX + lane * 128);
      unsigned long long t0 = __builtin_amdgcn_s_memrealtime();
      unsigned long long v;
      while (((v = __hip_atomic_load(slot, __ATOMIC_RELAXED, __HIP_MEMORY_SCOPE_SYSTEM)) >> 32) < call) {
        if (__hip_atomic_load(tmo, __ATOMIC_RELAXED, __HIP_MEMORY_SCOPE_AGENT) == call ||
            __builtin_amdgcn_s_memrealtime() - t0 > DEADLINE) {
          __hip_atomic_store(tmo, call, __ATOMIC_RELAXED, __HIP_MEMORY_SCOPE_AGENT);
          ok = false;
          break;
        }
        __builtin_amdgcn_s_sleep(1);
      }
      gm = ok ? __uint_as_float((unsigned)v) : 0.f;
    }
#pragma unroll
    for (int o = 4; o >= 1; o >>= 1) gm = fmaxf(gm, __shfl_xor(gm, o));
    if (lane == 0) {
      bool bad = __hip_atomic_load(tmo, __ATOMIC_RELAXED, __HIP_MEMORY_SCOPE_AGENT) == call;
      sred[8] = gm;
      sred[9] = bad ? 1.f : 0.f;
    }
  }
  __syncthreads();
  TS(62);
  const float gmax = sred[8];
  const bool bad = sred[9] != 0.f;
  const float scale = gmax / 448.f;

  unsigned char* qR = (unsigned char*)smem;
  unsigned char* qL = qR + BM * QS;
  quant_to_lds(accR, scale, qR, wr, wc, fr, fq);
  if (hasloc) quant_to_lds(accL, scale, qL, wr, wc, fr, fq);
  __syncthreads();
  {
    const int r = tid >> 2, sg = tid & 3;
    const u32x4 nanv = u32x4{0x7f7f7f7fu, 0x7f7f7f7fu, 0x7f7f7f7fu, 0x7f7f7f7fu};
    u32x4 v = *(const u32x4*)(qR + r * QS + sg * 16);
    if (bad) v = nanv;
    *(u32x4*)(a.out + (size_t)(srcr * MSH + arowR + r) * NSH + bcolR + sg * 16) = v;
    if (hasloc) {
      u32x4 v2 = *(const u32x4*)(qL + r * QS + sg * 16);
      if (bad) v2 = nanv;
      *(u32x4*)(a.out + (size_t)(rank * MSH + arowL + r) * NSH + bcolL + sg * 16) = v2;
    }
  }
  TS(63);
  if (tid == 0) __hip_atomic_fetch_add(cnt, 1u, __ATOMIC_RELAXED, __HIP_MEMORY_SCOPE_AGENT);
}

}

extern "C" void kernel_launch(int rank, int nranks, ncclComm_t comm, hipStream_t stream,
                              void* const* d_in, const int* in_sizes, int n_in, void* d_out, int out_size,
                              const int* in_dtypes, int out_dtype, void* const* d_out_all,
                              void* const* d_ws_all, size_t ws_bytes) {
  if (nranks != NR || ws_bytes < WS_NEED || (unsigned)rank >= (unsigned)NR) {
    fprintf(stderr, "unexpected configuration\n");
    return;
  }
  Args a;
  a.x = (const unsigned short*)d_in[0];
  a.w = (const unsigned short*)d_in[1];
  a.out = (unsigned char*)d_out;
  for (int r = 0; r < NR; ++r) a.ws[r] = (char*)d_ws_all[r];
  a.rank = rank;
  HIP_CHECK(hipFuncSetAttribute((const void*)ag_gemm_q, hipFuncAttributeMaxDynamicSharedMemorySize, LDS_BYTES));
  hipLaunchKernelGGL(ag_gemm_q, dim3(NWG), dim3(NTHR), LDS_BYTES, stream, a);
  HIP_CHECK(hipGetLastError());
}
```

```cpp
#include <hip/hip_runtime.h>
#include <rccl/rccl.h>
#include <cstdio>
#include <cstdint>

typedef __attribute__((ext_vector_type(8))) short bf16x8;
typedef __attribute__((ext_vector_type(4))) short s16x4;
typedef __attribute__((ext_vector_type(4))) float f32x4;
typedef __attribute__((ext_vector_type(4))) unsigned u32x4;

#define HIP_CHECK(expr) do { hipError_t _e = (expr); if (_e != hipSuccess) { \
    fprintf(stderr, "HIP error %s at %s:%d\n", hipGetErrorString(_e), __FILE__, __LINE__); \
    return; } } while (0)

namespace {

#define KS_ 128
constexpr int NR = 8;
constexpr int KK = 4096;
constexpr int NSH = 512;
constexpr int MSH = 512;
constexpr int XR = 256;
constexpr int WC = 128;
constexpr int BM = 128, BK = 64;
constexpr int NTHR = 512;
constexpr int KS = KS_;
constexpr int NSTAGE = 2 * (KS_ / 64);
constexpr int NSL = KK / KS;
constexpr int TPS = KS / BK;
constexpr int NT = KK / BK;
constexpr int NPUSH = 32;
constexpr int NDBL = 32;
constexpr int NG = 224;
constexpr int NWG = NPUSH + NG;
constexpr int NTILE = 256;

constexpr size_t OFF_CNT = 0;
constexpr size_t OFF_TMO = 128;
constexpr size_t OFF_DONE = 512;
constexpr size_t OFF_AMAX = 1024;
constexpr size_t OFF_GRAN = 2048;
constexpr size_t OFF_FLAG = 4096;
constexpr size_t OFF_XS = 1u << 20;
constexpr size_t XSLOT = (size_t)XR * KK * 2;
constexpr size_t OFF_WS = OFF_XS + NR * XSLOT;
constexpr size_t WSLOT = (size_t)KK * WC * 2;
constexpr size_t WS_NEED = OFF_WS + NR * WSLOT;
static_assert(OFF_FLAG + (size_t)NR * NSL * 128 <= (256u << 10), "flags");

#define TS(i) do {} while (0)
constexpr unsigned long long DEADLINE = 50000000ull;

constexpr int A_BYTES = BM * BK * 2;
constexpr int B_BYTES = BK * 64 * 2;
constexpr int LDS_MISC = NSTAGE * (A_BYTES + 2 * B_BYTES);
constexpr int LDS_BYTES = LDS_MISC + 64;
static_assert(LDS_BYTES > 81920 && LDS_BYTES <= 163840, "one workgroup per CU");

struct Args {
  const unsigned short* x;
  const unsigned short* w;
  unsigned char* out[NR];
  char* ws[NR];
  int rank;
};

__device__ __forceinline__ int lds_byte(int r, int c) {
  int st = (r >> 4) * 2 + (c >> 5), rr = r & 15, cc = c & 31, ob = rr * 64 + cc * 2;
  return st * 1024 + (ob ^ (((ob >> 9) & 1) << 5));
}
__device__ __forceinline__ void stage_rc(int b, int& R, int& C) {
  int st = b / 1024, sb = b % 1024, swz = sb ^ (((sb >> 9) & 1) << 5);
  R = (st >> 1) * 16 + swz / 64; C = (st & 1) * 32 + (swz % 64) / 2;
}
__device__ __forceinline__ int bswz(int row) { return ((row >> 1) & 1) | (((row >> 3) & 1) << 1); }

__device__ __forceinline__ void wait_u32(const unsigned* flag, unsigned target, unsigned* tmo, unsigned call) {
  unsigned long long t0 = __builtin_amdgcn_s_memrealtime();
  while (__hip_atomic_load(flag, __ATOMIC_RELAXED, __HIP_MEMORY_SCOPE_SYSTEM) < target) {
    if (__hip_atomic_load(tmo, __ATOMIC_RELAXED, __HIP_MEMORY_SCOPE_AGENT) == call ||
        __builtin_amdgcn_s_memrealtime() - t0 > DEADLINE) {
      __hip_atomic_store(tmo, call, __ATOMIC_RELAXED, __HIP_MEMORY_SCOPE_AGENT);
      return;
    }
    __builtin_amdgcn_s_sleep(2);
  }
}
__device__ __forceinline__ void store16_sys(void* p, u32x4 v) {
  asm volatile("global_store_dwordx4 %0, %1, off sc0 sc1\n\ts_nop 1" :: "v"(p), "v"(v) : "memory");
}

template <int BNT>
__device__ __forceinline__ void gemm_role(const Args& a, char* smem, char* myws, unsigned call,
                                          int oa, int ha, int ob, int hb) {
  constexpr int NFR = BNT / 32;
  constexpr int NB = BNT / 64;
  constexpr int BUF = A_BYTES + NB * B_BYTES;
  constexpr int NLD = 2 + NB;
  const int tid = threadIdx.x;
  const int rank = a.rank, ri = rank >> 2, rj = rank & 3;
  unsigned* tmo = (unsigned*)(myws + OFF_TMO);
  const int wid = tid >> 6, lane = tid & 63, wr = wid >> 1, wc = wid & 1, fr = lane & 15, fq = lane >> 4;

  const unsigned short* Abase = (const unsigned short*)(myws + OFF_XS + (size_t)oa * XSLOT) + (size_t)(ha * BM) * KK;
  const unsigned short* Bbase = (const unsigned short*)(myws + OFF_WS + (size_t)ob * WSLOT) + hb * 64;
  const unsigned short *a0, *a1, *b0;
  { int R, C; stage_rc(tid * 16, R, C); a0 = Abase + R * KK + C;
    stage_rc(tid * 16 + 8192, R, C); a1 = Abase + R * KK + C; }
  { int b = tid * 16, row = b >> 7, sl = (b & 127) >> 4;
    b0 = Bbase + row * WC + (((sl >> 1) ^ bswz(row)) << 4) + ((sl & 1) << 3); }
  int aoff[2][2], boff[NFR][2][2];
#pragma unroll
  for (int m = 0; m < 2; ++m)
#pragma unroll
    for (int kk = 0; kk < 2; ++kk) aoff[m][kk] = lds_byte(wr * 32 + m * 16 + fr, kk * 32 + fq * 8);
#pragma unroll
  for (int n = 0; n < NFR; ++n)
#pragma unroll
    for (int kk = 0; kk < 2; ++kk)
#pragma unroll
      for (int t = 0; t < 2; ++t) {
        const int co = wc * (BNT / 2) + n * 16, sub = co >> 6, ch = (co & 63) >> 4;
        const int row = kk * 32 + 8 * fq + 4 * t + (fr >> 2);
        boff[n][kk][t] = A_BYTES + sub * B_BYTES + row * 128 + ((ch ^ bswz(row)) << 5) + ((lane & 3) << 3);
      }
  const unsigned lds0 = (unsigned)(size_t)smem;
  const unsigned* flagA = (const unsigned*)(myws + OFF_FLAG + (size_t)oa * NSL * 128);
  const unsigned* flagB = (const unsigned*)(myws + OFF_FLAG + (size_t)ob * NSL * 128);
  const unsigned ftarget = call * NPUSH;

  auto stage = [&](int t, int buf) {
    char* sa = smem + buf * BUF;
    __builtin_amdgcn_global_load_lds((const unsigned*)(a0 + t * BK), (unsigned*)(sa + tid * 16), 16, 0, 17);
    __builtin_amdgcn_global_load_lds((const unsigned*)(a1 + t * BK), (unsigned*)(sa + tid * 16 + 8192), 16, 0, 17);
#pragma unroll
    for (int u = 0; u < NB; ++u)
      __builtin_amdgcn_global_load_lds((const unsigned*)(b0 + (size_t)t * BK * WC + u * 64), (unsigned*)(sa + A_BYTES + u * B_BYTES + tid * 16), 16, 0, 17);
  };
  auto stage_slice = [&](int s) {
#pragma unroll
    for (int u = 0; u < TPS; ++u) stage(s * TPS + u, (s & 1) * TPS + u);
  };
  auto wait_slice = [&](int s) {
    if (tid == 0) { wait_u32(flagA + s * 32, ftarget, tmo, call); wait_u32(flagB + s * 32, ftarget, tmo, call); }
    __builtin_amdgcn_s_barrier();
  };
  const unsigned* fptr = (lane & 1) ? flagB : flagA;
  unsigned fv = 0;
  auto flag_load = [&](int s) {
    asm volatile("global_load_dword %0, %1, off sc0 sc1" : "=v"(fv) : "v"(fptr + s * 32) : "memory");
  };
  volatile int* status = (volatile int*)(smem + LDS_MISC + 48);

  f32x4 acc[2][NFR];
#pragma unroll
  for (int m = 0; m < 2; ++m)
#pragma unroll
    for (int n = 0; n < NFR; ++n) acc[m][n] = f32x4{0.f, 0.f, 0.f, 0.f};

  auto compute_tile = [&](int buf) {
    const unsigned lbase = lds0 + buf * BUF;
    bf16x8 af[2][2];
    s16x4 bl[NFR][2][2];
#pragma unroll
    for (int kk = 0; kk < 2; ++kk) {
#pragma unroll
      for (int m = 0; m < 2; ++m)
        asm volatile("ds_read_b128 %0, %1" : "=v"(af[m][kk]) : "v"(lbase + aoff[m][kk]) : "memory");
#pragma unroll
      for (int n = 0; n < NFR; ++n)
#pragma unroll
        for (int h = 0; h < 2; ++h)
          asm volatile("ds_read_b64_tr_b16 %0, %1" : "=v"(bl[n][kk][h]) : "v"(lbase + boff[n][kk][h]) : "memory");
    }
#pragma unroll
    for (int kk = 0; kk < 2; ++kk) {
      if (kk == 0) asm volatile("s_waitcnt lgkmcnt(%0)" :: "n"(2 + 2 * NFR) : "memory");
      else asm volatile("s_waitcnt lgkmcnt(0)" ::: "memory");
      asm volatile("" : "+v"(af[0][kk]), "+v"(af[1][kk]) :: "memory");
#pragma unroll
      for (int n = 0; n < NFR; ++n) asm volatile("" : "+v"(bl[n][kk][0]), "+v"(bl[n][kk][1]) :: "memory");
      __builtin_amdgcn_sched_barrier(0);
      __builtin_amdgcn_s_setprio(1);
#pragma unroll
      for (int n = 0; n < NFR; ++n) {
        const s16x4 lo = bl[n][kk][0], hi = bl[n][kk][1];
        const bf16x8 bf = bf16x8{lo[0], lo[1], lo[2], lo[3], hi[0], hi[1], hi[2], hi[3]};
#pragma unroll
        for (int m = 0; m < 2; ++m)
          acc[m][n] = __builtin_amdgcn_mfma_f32_16x16x32_bf16(af[m][kk], bf, acc[m][n], 0, 0, 0);
      }
      __builtin_amdgcn_s_setprio(0);
      __builtin_amdgcn_sched_barrier(0);
    }
  };

  const bool w0 = __builtin_amdgcn_readfirstlane(wid) == 0;
  wait_slice(0);
  TS(1);
  stage_slice(0);
  if (w0) flag_load(1);
  for (int s = 0; s < NSL; ++s) {
    asm volatile("s_waitcnt vmcnt(0)" : "+v"(fv) :: "memory");
    if (w0) {
      const bool ok = __all(fv >= ftarget);
      if (lane == 0) status[s & 1] = ok ? 1 : 0;
      asm volatile("s_waitcnt lgkmcnt(0)" ::: "memory");
    }
    __builtin_amdgcn_s_barrier();
    const bool more = s + 1 < NSL;
    const bool ready = __builtin_amdgcn_readfirstlane(status[s & 1]) != 0;
    if (more && ready) {
      TS(2 + s);
      stage_slice(s + 1);
      if (w0 && s + 2 < NSL) flag_load(s + 2);
    }
#pragma unroll
    for (int u = 0; u < TPS; ++u) compute_tile((s & 1) * TPS + u);
    if (more && !ready) {
      wait_slice(s + 1);
      TS(2 + s);
      stage_slice(s + 1);
      if (w0 && s + 2 < NSL) flag_load(s + 2);
    }
  }
  TS(70);

  float mx = 0.f;
#pragma unroll
  for (int m = 0; m < 2; ++m)
#pragma unroll
    for (int n = 0; n < NFR; ++n)
#pragma unroll
      for (int j = 0; j < 4; ++j) { float v = fmaxf(acc[m][n][j], 0.f); acc[m][n][j] = v; mx = fmaxf(mx, v); }
#pragma unroll
  for (int o = 32; o >= 1; o >>= 1) mx = fmaxf(mx, __shfl_xor(mx, o));
  float* sred = (float*)(smem + LDS_MISC);
  if (lane == 0) sred[wid] = mx;
  __syncthreads();
  if (tid == 0) {
    float m8 = 0.f;
#pragma unroll
    for (int i = 0; i < 8; ++i) m8 = fmaxf(m8, sred[i]);
    unsigned long long key = ((unsigned long long)call << 32) | __float_as_uint(m8);
    __hip_atomic_store((unsigned long long*)(myws + OFF_GRAN) + (blockIdx.x - NPUSH), key, __ATOMIC_RELAXED,
                       __HIP_MEMORY_SCOPE_AGENT);
  }
  TS(71);
  if (wid == 0) {
    float gm = 0.f;
    bool ok = true;
    if (lane < NR) {
      const unsigned long long* slot = (const unsigned long long*)(myws + OFF_AMAX + lane * 128);
      unsigned long long t0 = __builtin_amdgcn_s_memrealtime();
      unsigned long long v;
      while (((v = __hip_atomic_load(slot, __ATOMIC_RELAXED, __HIP_MEMORY_SCOPE_SYSTEM)) >> 32) < call) {
        if (__hip_atomic_load(tmo, __ATOMIC_RELAXED, __HIP_MEMORY_SCOPE_AGENT) == call ||
            __builtin_amdgcn_s_memrealtime() - t0 > DEADLINE) {
          __hip_atomic_store(tmo, call, __ATOMIC_RELAXED, __HIP_MEMORY_SCOPE_AGENT);
          ok = false;
          break;
        }
        __builtin_amdgcn_s_sleep(1);
      }
      gm = ok ? __uint_as_float((unsigned)v) : 0.f;
    }
#pragma unroll
    for (int o = 4; o >= 1; o >>= 1) gm = fmaxf(gm, __shfl_xor(gm, o));
    if (lane == 0) {
      bool bad = __hip_atomic_load(tmo, __ATOMIC_RELAXED, __HIP_MEMORY_SCOPE_AGENT) == call;
      sred[8] = gm;
      sred[9] = bad ? 1.f : 0.f;
    }
  }
  __syncthreads();
  TS(72);
  const float gmax = sred[8];
  const bool bad = sred[9] != 0.f;
  const float scale = gmax / 448.f;

  constexpr int QS = BNT + 16;
  unsigned char* q = (unsigned char*)smem;
#pragma unroll
  for (int m = 0; m < 2; ++m)
#pragma unroll
    for (int n = 0; n < NFR; ++n) {
      float v0 = acc[m][n][0] / scale, v1 = acc[m][n][1] / scale, v2 = acc[m][n][2] / scale, v3 = acc[m][n][3] / scale;
      v0 = fminf(v0, 448.f); v1 = fminf(v1, 448.f); v2 = fminf(v2, 448.f); v3 = fminf(v3, 448.f);
      int p01 = __builtin_amdgcn_cvt_pk_fp8_f32(v0, v1, 0, false);
      int p23 = __builtin_amdgcn_cvt_pk_fp8_f32(v2, v3, 0, false);
      int r = wr * 32 + m * 16 + fq * 4, cc = wc * (BNT / 2) + n * 16 + fr;
      q[(r + 0) * QS + cc] = (unsigned char)(p01 & 0xff);
      q[(r + 1) * QS + cc] = (unsigned char)((p01 >> 8) & 0xff);
      q[(r + 2) * QS + cc] = (unsigned char)(p23 & 0xff);
      q[(r + 3) * QS + cc] = (unsigned char)((p23 >> 8) & 0xff);
    }
  __syncthreads();
  {
    unsigned char* obase = a.out[ob] + (size_t)(oa * MSH + ri * XR + ha * BM) * NSH + rj * WC + hb * 64;
    const u32x4 nanv = u32x4{0x7f7f7f7fu, 0x7f7f7f7fu, 0x7f7f7f7fu, 0x7f7f7f7fu};
    constexpr int CPR = BNT / 16;
#pragma unroll
    for (int it = 0; it < NB; ++it) {
      const int idx = tid + it * NTHR, r = idx / CPR, sg = idx % CPR;
      u32x4 v = *(const u32x4*)(q + r * QS + sg * 16);
      if (bad) v = nanv;
      store16_sys(obase + (size_t)r * NSH + sg * 16, v);
    }
    asm volatile("s_waitcnt vmcnt(0)" ::: "memory");
    __syncthreads();
    if (tid == 0)
      __hip_atomic_fetch_add((unsigned*)(a.ws[ob] + OFF_DONE), (unsigned)NB, __ATOMIC_RELAXED, __HIP_MEMORY_SCOPE_SYSTEM);
  }
  TS(73);
}

template <int NX, int NW, int XT, int WT>
__device__ __forceinline__ void push_loop(const Args& a, char* myws, int xt, int wt) {
  constexpr int LPX = KS * 2 / 16;
  constexpr int NLD = NX + NW, NST = 4 * NX + 2 * NW;
  const int tid = threadIdx.x, rank = a.rank, q = blockIdx.x;
  const int ix = q >> 4;
  const char* xsrc; char* xdst[4];
  { int r = q * 16 + xt / LPX, l = xt % LPX;
    xsrc = (const char*)a.x + (size_t)r * (KK * 2) + l * 16;
    size_t xdo = OFF_XS + (size_t)rank * XSLOT + (size_t)(r - ix * XR) * (KK * 2) + l * 16;
#pragma unroll
    for (int d = 0; d < 4; ++d) xdst[d] = a.ws[ix * 4 + ((rank + d) & 3)] + xdo; }
  const char* wsrc; char* wdst[2];
  { int kr = q * (KS / 32) + wt / 64, c = wt % 64, jw = c >> 4;
    wsrc = (const char*)a.w + (size_t)kr * (NSH * 2) + c * 16;
    size_t wdo = OFF_WS + (size_t)rank * WSLOT + (size_t)kr * (WC * 2) + (c & 15) * 16;
    wdst[0] = a.ws[jw] + wdo; wdst[1] = a.ws[4 + jw] + wdo; }
  unsigned* fl = (unsigned*)(a.ws[tid & 7] + OFF_FLAG + (size_t)rank * NSL * 128);

  u32x4 va[NLD], vb[NLD];
  auto ld = [&](u32x4 (&d)[NLD], int sl) {
#pragma unroll
    for (int i = 0; i < NX; ++i) {
      const char* sp = xsrc + (size_t)i * (XT / LPX) * (KK * 2) + (size_t)sl * (KS * 2);
      asm volatile("global_load_dwordx4 %0, %1, off" : "=v"(d[i]) : "v"(sp) : "memory");
    }
#pragma unroll
    for (int i = 0; i < NW; ++i) {
      const char* sp = wsrc + (size_t)i * (WT / 64) * (NSH * 2) + (size_t)sl * KS * (NSH * 2);
      asm volatile("global_load_dwordx4 %0, %1, off" : "=v"(d[NX + i]) : "v"(sp) : "memory");
    }
  };
  auto body = [&](int sl, u32x4 (&cur)[NLD], u32x4 (&nxt)[NLD]) {
    ld(nxt, (sl + 1 < NSL) ? sl + 1 : sl);
    if (sl == 0) asm volatile("s_waitcnt vmcnt(%0)" :: "n"(NLD) : "memory");
    else asm volatile("s_waitcnt vmcnt(%0)" :: "n"(NLD + NST) : "memory");
#pragma unroll
    for (int i = 0; i < NLD; ++i) asm volatile("" : "+v"(cur[i]) :: "memory");
#pragma unroll
    for (int i = 0; i < NX; ++i)
#pragma unroll
      for (int d = 0; d < 4; ++d)
        store16_sys(xdst[d] + (size_t)i * (XT / LPX) * (KK * 2) + (size_t)sl * (KS * 2), cur[i]);
#pragma unroll
    for (int i = 0; i < NW; ++i)
#pragma unroll
      for (int d = 0; d < 2; ++d)
        store16_sys(wdst[d] + (size_t)i * (WT / 64) * (WC * 2) + (size_t)sl * KS * (WC * 2), cur[NX + i]);
    if (sl > 0) {
      asm volatile("s_waitcnt vmcnt(%0)" :: "n"(NLD + NST) : "memory");
      __builtin_amdgcn_s_barrier();
      if (tid < NR) __hip_atomic_fetch_add(fl + (sl - 1) * 32, 1u, __ATOMIC_RELAXED, __HIP_MEMORY_SCOPE_SYSTEM);
      TS(sl);
    }
  };
  ld(va, 0);
  for (int sl = 0; sl < NSL; sl += 2) { body(sl, va, vb); body(sl + 1, vb, va); }
  asm volatile("s_waitcnt vmcnt(0)" ::: "memory");
  __builtin_amdgcn_s_barrier();
  if (tid < NR) __hip_atomic_fetch_add(fl + (NSL - 1) * 32, 1u, __ATOMIC_RELAXED, __HIP_MEMORY_SCOPE_SYSTEM);
  TS(NSL);
}
__device__ __forceinline__ void push_role(const Args& a, char* myws) {
  static_assert(NSL % 2 == 0 && (KS == 128 || KS % 256 == 0), "push shape");
  const int tid = threadIdx.x;
  if constexpr (KS == 128) {
    if (__builtin_amdgcn_readfirstlane(tid) < 256) push_loop<1, 0, 256, 256>(a, myws, tid, 0);
    else push_loop<0, 1, 256, 256>(a, myws, 0, tid - 256);
  } else {
    push_loop<KS / 256, KS / 256, NTHR, NTHR>(a, myws, tid, tid);
  }
}

__global__ __launch_bounds__(NTHR, 2) void ag_gemm_q(Args a) {
  extern __shared__ __attribute__((aligned(16))) char smem[];
  const int tid = threadIdx.x;
  const int rank = a.rank;
  char* myws = a.ws[rank];
  unsigned* cnt = (unsigned*)(myws + OFF_CNT);
  unsigned* tmo = (unsigned*)(myws + OFF_TMO);
  const unsigned call = __hip_atomic_load(cnt, __ATOMIC_RELAXED, __HIP_MEMORY_SCOPE_AGENT) / NWG + 1;

  TS(0);
  if (blockIdx.x < NPUSH) {
    push_role(a, myws);
    if (blockIdx.x == 1 && tid < 64) {
      const unsigned long long* gr = (const unsigned long long*)(myws + OFF_GRAN);
      unsigned long long t0 = __builtin_amdgcn_s_memrealtime();
      unsigned mxb = 0;
      for (;;) {
        bool ok = true;
        mxb = 0;
#pragma unroll
        for (int k = 0; k < 4; ++k) {
          const int gi = tid + k * 64;
          if (gi < NG) {
            unsigned long long v = __hip_atomic_load(gr + gi, __ATOMIC_RELAXED, __HIP_MEMORY_SCOPE_AGENT);
            ok &= (unsigned)(v >> 32) >= call;
            mxb = max(mxb, (unsigned)v);
          }
        }
        if (__all(ok)) break;
        if (__hip_atomic_load(tmo, __ATOMIC_RELAXED, __HIP_MEMORY_SCOPE_AGENT) == call ||
            __builtin_amdgcn_s_memrealtime() - t0 > DEADLINE) {
          __hip_atomic_store(tmo, call, __ATOMIC_RELAXED, __HIP_MEMORY_SCOPE_AGENT);
          break;
        }
        __builtin_amdgcn_s_sleep(1);
      }
#pragma unroll
      for (int o = 32; o >= 1; o >>= 1) mxb = max(mxb, (unsigned)__shfl_xor((int)mxb, o));
      if (tid < NR)
        __hip_atomic_store((unsigned long long*)(a.ws[tid] + OFF_AMAX + rank * 128),
                           ((unsigned long long)call << 32) | mxb, __ATOMIC_RELAXED, __HIP_MEMORY_SCOPE_SYSTEM);
    }
    if (blockIdx.x == 0 && tid == 0) {
      wait_u32((const unsigned*)(myws + OFF_DONE), call * NTILE, tmo, call);
      if (__hip_atomic_load(tmo, __ATOMIC_RELAXED, __HIP_MEMORY_SCOPE_AGENT) == call) a.out[rank][0] = 0x7f;
      TS(74);
    }
    if (tid == 0) __hip_atomic_fetch_add(cnt, 1u, __ATOMIC_RELAXED, __HIP_MEMORY_SCOPE_AGENT);
    return;
  }

  const int g = blockIdx.x - NPUSH;
  const int order = (g & 7) * (NG / 8) + (g >> 3);
  if (order < NDBL) {
    const int rt = order >> 3;
    gemm_role<128>(a, smem, myws, call, rt >> 1, rt & 1, order & 7, 0);
  } else {
    const int t = order - NDBL, rt = NDBL / 8 + (t >> 4), ct = t & 15;
    gemm_role<64>(a, smem, myws, call, rt >> 1, rt & 1, ct >> 1, ct & 1);
  }
  if (tid == 0) __hip_atomic_fetch_add(cnt, 1u, __ATOMIC_RELAXED, __HIP_MEMORY_SCOPE_AGENT);
}

}

extern "C" void kernel_launch(int rank, int nranks, ncclComm_t comm, hipStream_t stream,
                              void* const* d_in, const int* in_sizes, int n_in, void* d_out, int out_size,
                              const int* in_dtypes, int out_dtype, void* const* d_out_all,
                              void* const* d_ws_all, size_t ws_bytes) {
  if (nranks != NR || ws_bytes < WS_NEED || (unsigned)rank >= (unsigned)NR) {
    fprintf(stderr, "unexpected configuration\n");
    return;
  }
  Args a;
  a.x = (const unsigned short*)d_in[0];
  a.w = (const unsigned short*)d_in[1];
  for (int r = 0; r < NR; ++r) { a.ws[r] = (char*)d_ws_all[r]; a.out[r] = (unsigned char*)d_out_all[r]; }
  a.rank = rank;
  HIP_CHECK(hipFuncSetAttribute((const void*)ag_gemm_q, hipFuncAttributeMaxDynamicSharedMemorySize, LDS_BYTES));
  hipLaunchKernelGGL(ag_gemm_q, dim3(NWG), dim3(NTHR), LDS_BYTES, stream, a);
  HIP_CHECK(hipGetLastError());
}
```
